```python
import numpy as np

kernel_name = "gpu_gemm_a2a_m4096_k4096_n4096_w8a8_dqrelu"


ABSMAX_RTOL = 0.001

COMPUTE_DTYPE = "int8"
EPILOGUE = "dequant_relu"
IN_QUANT = "int8"
OUT_QUANT = None
IN_DTYPES = {"x": "int8", "w_mat": "int8",
             "scale_x": "float32", "scale_w": "float32"}
WORLD_SIZE = 8

def setup_inputs():

    x = np.random.randint(-127, 128, size=(4096, 4096)).astype(np.int8)
    w_mat = np.random.randint(-127, 128, size=(4096, 4096)).astype(np.int8)
    scale_x = (0.02 * (1.0 + np.random.rand(1))).astype(np.float32)
    scale_w = (0.02 * (1.0 + np.random.rand(1))).astype(np.float32)
    return {"x": x, "w_mat": w_mat, "scale_x": scale_x, "scale_w": scale_w}

def reference(x, w_mat, scale_x, scale_w):
    acc = x.astype(np.float64) @ w_mat.astype(np.float64)
    y = (acc.astype(np.float32) * (scale_x * scale_w)).astype(np.float32)
    return np.maximum(y, 0.0).astype(np.float32)
```

```cpp
#include <hip/hip_runtime.h>
#include <rccl/rccl.h>
#include <cstdio>

#define HIP_CHECK(expr) do { hipError_t _e = (expr); if (_e != hipSuccess) { \
    fprintf(stderr, "HIP error %s at %s:%d\n", hipGetErrorString(_e), __FILE__, __LINE__); \
    return; } } while (0)

namespace {
typedef unsigned u32;
typedef unsigned long long u64;
typedef int i32x4 __attribute__((ext_vector_type(4)));
typedef int i32x2 __attribute__((ext_vector_type(2)));
typedef float f32x4 __attribute__((ext_vector_type(4)));
typedef __attribute__((address_space(3))) void* lds_vp;
typedef __attribute__((address_space(3))) i32x2* lds_i2p;
typedef __attribute__((address_space(3))) i32x4* lds_i4p;
typedef __attribute__((address_space(3))) char* lds_cp;

constexpr int NR = 8;
constexpr int MS = 512;
constexpr int KD = 4096;
constexpr int ND = 4096;
constexpr int NS = 512;
constexpr int BK = 128;
constexpr int NSTEP = KD / BK;
constexpr int NWG = 256;
constexpr int NTHR = 512;
constexpr u64 DEADLINE = 50000000ull;

struct Args {
  const char* x; const char* w; const float* sx; const float* sw;
  float* out[NR];
  u32* ws[NR];
  int rank;
};

__device__ __forceinline__ i32x4 make_srd(const void* p, u32 bytes) {
  const u64 a = (u64)p;
  return i32x4{(int)(u32)a, (int)((u32)(a >> 32) & 0xffffu), (int)bytes, 0x00020000};
}
__device__ __forceinline__ void dma16(i32x4 srd, u32 lds_addr, int voff, int soff) {
  u32 keep;
  asm volatile("s_mov_b32 %0, m0\n\ts_mov_b32 m0, %1\n\ts_nop 0\n\tbuffer_load_dwordx4 %2, %3, %4 offen lds\n\ts_mov_b32 m0, %0"
               : "=&s"(keep) : "s"(lds_addr), "v"(voff), "s"(srd), "s"(soff) : "memory");
}
template <int N> __device__ __forceinline__ void wait_vm() { asm volatile("s_waitcnt vmcnt(%0)" :: "i"(N) : "memory"); }

template <int I> struct IC { static constexpr int value = I; };
template <int N, int I = 0, class F> __device__ __forceinline__ void static_for(F&& f) {
  if constexpr (I < N) { f(IC<I>{}); static_for<N, I + 1>(f); }
}

template <int MT, int NSTG>
__device__ __forceinline__ void gemm_tile(const Args& a, lds_cp smem, int row0, int col0, int wave, int lane, i32x4 (&acc)[MT][2]) {
  constexpr int TM = 32 * MT;
  constexpr int XI = (TM / 8 + 7) / 8;
  constexpr int WB = BK * 128;
  constexpr int XB = XI * 8 * 1024;
  constexpr int SB = WB + XB;
  constexpr int PER = 2 + XI;
  const i32x4 rW = make_srd(a.w, (u32)KD * ND);
  const i32x4 rX = make_srd(a.x, (u32)MS * KD);
  const u32 sm0 = (u32)(size_t)smem;
  const int wsig = ((lane >> 4) & 3) | ((wave & 1) << 2);
  const int w_voff = (wave * 16 + (lane >> 3)) * ND + col0 + 16 * ((lane & 7) ^ wsig);
  int x_voff[XI];
#pragma unroll
  for (int j = 0; j < XI; ++j) {
    const int i = wave + 8 * j;
    const int xsig = (4 * (i & 1) + (lane >> 4)) & 7;
    x_voff[j] = (i * 8 < TM) ? (row0 + 8 * i + (lane >> 3)) * KD + 16 * ((lane & 7) ^ xsig) : (int)0x7fffff00;
  }
  auto stage = [&](int t, int slot) {
    const u32 base = sm0 + slot * SB;
    dma16(rW, base + wave * 2048, w_voff, t * BK * ND);
    dma16(rW, base + wave * 2048 + 1024, w_voff, t * BK * ND + 8 * ND);
#pragma unroll
    for (int j = 0; j < XI; ++j) dma16(rX, base + WB + (wave + 8 * j) * 1024, x_voff[j], t * BK);
  };
  const int g = lane >> 4, q = (lane & 15) >> 1, p = lane & 1;
  const int rsig = (q >> 1) | ((g & 1) << 2);
  int w_rd[2];
#pragma unroll
  for (int nt = 0; nt < 2; ++nt) w_rd[nt] = (16 * g + q) * 128 + 16 * (((wave & 3) * 2 + nt) ^ rsig) + 8 * p;
  const int m = lane & 15;
  int x_rd[2];
#pragma unroll
  for (int s = 0; s < 2; ++s) x_rd[s] = WB + ((wave >> 2) * MT * 16 + m) * 128 + 16 * ((4 * s + g) ^ ((m >> 1) & 7));
#pragma unroll
  for (int i = 0; i < MT; ++i) { acc[i][0] = i32x4{0, 0, 0, 0}; acc[i][1] = i32x4{0, 0, 0, 0}; }

#pragma unroll
  for (int t = 0; t < NSTG - 1; ++t) stage(t, t);
  int slot = 0, fslot = NSTG - 1;
  auto compute = [&](int sl) {
    lds_cp base = smem + sl * SB;
#pragma unroll
    for (int s = 0; s < 2; ++s) {
      i32x4 wf[2], xf[MT];
#pragma unroll
      for (int nt = 0; nt < 2; ++nt) {
        i32x2 lo = __builtin_amdgcn_ds_read_tr8_b64_v2i32((lds_i2p)(base + w_rd[nt] + s * 64 * 128));
        i32x2 hi = __builtin_amdgcn_ds_read_tr8_b64_v2i32((lds_i2p)(base + w_rd[nt] + s * 64 * 128 + 8 * 128));
        wf[nt] = i32x4{lo[0], lo[1], hi[0], hi[1]};
      }
#pragma unroll
      for (int i = 0; i < MT; ++i) xf[i] = *(lds_i4p)(base + x_rd[s] + i * 16 * 128);
#pragma unroll
      for (int i = 0; i < MT; ++i)
#pragma unroll
        for (int nt = 0; nt < 2; ++nt)
          acc[i][nt] = __builtin_amdgcn_mfma_i32_16x16x64_i8(wf[nt], xf[i], acc[i][nt], 0, 0, 0);
    }
  };
  auto sync = [&]() {
    asm volatile("s_waitcnt lgkmcnt(0)" ::: "memory");
    __builtin_amdgcn_s_barrier();
    asm volatile("" ::: "memory");
  };
#pragma unroll 1
  for (int t = 0; t < NSTEP - (NSTG - 1); ++t) {
    wait_vm<(NSTG - 2) * PER>();
    sync();
    stage(t + NSTG - 1, fslot);
    compute(slot);
    slot = slot + 1 == NSTG ? 0 : slot + 1;
    fslot = fslot + 1 == NSTG ? 0 : fslot + 1;
  }
  static_for<NSTG - 1>([&](auto J) {
    wait_vm<(NSTG - 2 - J.value) * PER>();
    sync();
    compute(slot);
    slot = slot + 1 == NSTG ? 0 : slot + 1;
  });
}

__device__ __forceinline__ float epi(int acc, float s) { float y = (float)acc * s; return (y <= 0.f) ? 0.f : y; }

__global__ void __launch_bounds__(NTHR, 2) fused_kernel(Args a) {
  extern __shared__ __attribute__((aligned(16))) char smem_raw[];
  lds_cp smem = (lds_cp)smem_raw;
  const int lane = threadIdx.x & 63;
  const int wave = __builtin_amdgcn_readfirstlane(threadIdx.x >> 6);
  const int bid = blockIdx.x;
  const int xcd = bid & 7, idx = bid >> 3;
  const int strip = (xcd * 4 + (idx >> 3) + a.rank * 4) & 31;
  const int dest = strip >> 2;
  const int col0 = strip * 128;
  const int row0 = (idx & 7) * 64;
  u32* myws = a.ws[a.rank];
  u32 call = 0;
  if (bid == 0 && threadIdx.x == 0) {
    call = __hip_atomic_load(myws, __ATOMIC_RELAXED, __HIP_MEMORY_SCOPE_AGENT) + 1;
    __hip_atomic_store(myws, call, __ATOMIC_RELAXED, __HIP_MEMORY_SCOPE_AGENT);
  }
  const float scale = a.sx[0] * a.sw[0];
  constexpr int MT = 2;
  i32x4 acc[MT][2];
  gemm_tile<MT, 6>(a, smem, row0, col0, wave, lane, acc);
  float* ob = a.out[(unsigned)dest < (unsigned)NR ? dest : 0];
  auto rO = __builtin_amdgcn_make_buffer_rsrc((void*)ob, 0, (u32)(NR * MS * NS * 4), 0x00020000);
#pragma unroll
  for (int i = 0; i < MT; ++i)
#pragma unroll
    for (int nt = 0; nt < 2; ++nt) {
      const int r = a.rank * MS + row0 + (wave >> 2) * MT * 16 + i * 16 + (lane & 15);
      const int c = (col0 & (NS - 1)) + (wave & 3) * 32 + nt * 16 + (lane >> 4) * 4;
      f32x4 v = {epi(acc[i][nt][0], scale), epi(acc[i][nt][1], scale), epi(acc[i][nt][2], scale), epi(acc[i][nt][3], scale)};
      __builtin_amdgcn_raw_buffer_store_b128(__builtin_bit_cast(i32x4, v), rO, (r * NS + c) * 4, 0, 17  );
    }
  wait_vm<0>();
  if (dest != a.rank && lane == 0)
    __hip_atomic_fetch_add(a.ws[dest] + 32 * (1 + a.rank), 1u, __ATOMIC_RELAXED, __HIP_MEMORY_SCOPE_SYSTEM);
  if (bid == 0 && wave == 0) {
    call = __builtin_amdgcn_readfirstlane(call);
    const u32 target = call * 256u;
    const u64 t0 = __builtin_amdgcn_s_memrealtime();
    const bool mine = lane < NR && lane != a.rank;
    bool tmo = false;
    for (;;) {
      u32 v = mine ? __hip_atomic_load(myws + 32 * (1 + lane), __ATOMIC_RELAXED, __HIP_MEMORY_SCOPE_SYSTEM) : target;
      if (__all((int)(v - target) >= 0)) break;
      if (__builtin_amdgcn_s_memrealtime() - t0 > DEADLINE) { tmo = true; break; }
      __builtin_amdgcn_s_sleep(1);
    }
    if (tmo && lane == 0) a.out[a.rank][(size_t)a.rank * MS * NS] = __builtin_nanf("");
  }
}
}

extern "C" void kernel_launch(int rank, int nranks, ncclComm_t comm, hipStream_t stream,
                              void* const* d_in, const int* in_sizes, int n_in,
                              void* d_out, int out_size, const int* in_dtypes, int out_dtype,
                              void* const* d_out_all, void* const* d_ws_all, size_t ws_bytes) {
  if (nranks != NR) return;
  constexpr int SMEM = 6 * (BK * 128 + 8 * 1024);
  static bool init[NR] = {};
  if ((unsigned)rank >= (unsigned)NR) return;
  if (!init[rank]) {
    HIP_CHECK(hipFuncSetAttribute((const void*)fused_kernel, hipFuncAttributeMaxDynamicSharedMemorySize, SMEM));
    init[rank] = true;
  }
  Args a;
  a.x = (const char*)d_in[0]; a.w = (const char*)d_in[1]; a.sx = (const float*)d_in[2]; a.sw = (const float*)d_in[3];
  for (int p = 0; p < NR; ++p) { a.out[p] = (float*)d_out_all[p]; a.ws[p] = (u32*)d_ws_all[p]; }
  a.rank = rank;
  fused_kernel<<<NWG, NTHR, SMEM, stream>>>(a);
}
```

```cpp
#include <hip/hip_runtime.h>
#include <rccl/rccl.h>
#include <cstdio>

#define HIP_CHECK(expr) do { hipError_t _e = (expr); if (_e != hipSuccess) { \
    fprintf(stderr, "HIP error %s at %s:%d\n", hipGetErrorString(_e), __FILE__, __LINE__); \
    return; } } while (0)

namespace {
typedef unsigned u32;
typedef unsigned long long u64;
typedef int i32x4 __attribute__((ext_vector_type(4)));
typedef int i32x2 __attribute__((ext_vector_type(2)));
typedef float f32x4 __attribute__((ext_vector_type(4)));
typedef __attribute__((address_space(3))) void* lds_vp;
typedef __attribute__((address_space(3))) i32x2* lds_i2p;
typedef __attribute__((address_space(3))) i32x4* lds_i4p;
typedef __attribute__((address_space(3))) char* lds_cp;

constexpr int NR = 8;
constexpr int MS = 512;
constexpr int KD = 4096;
constexpr int ND = 4096;
constexpr int NS = 512;
constexpr int BK = 128;
constexpr int NSTEP = KD / BK;
constexpr int NWG = 256;
constexpr int NTHR = 512;
#define NSTAGES 4
#define PACE 2
#define HDRSLEEP 8
#define OUT_AUX 17
constexpr int SMEM = (NSTAGES < 5 ? 5 : NSTAGES) * (BK * 128 + 8 * 1024) + 1024;
constexpr u64 DEADLINE = 50000000ull;

struct Args {
  const char* x; const char* w; const float* sx; const float* sw;
  float* out[NR];
  u32* ws[NR];
  int rank;
};

__device__ __forceinline__ i32x4 make_srd(const void* p, u32 bytes) {
  const u64 a = (u64)p;
  return i32x4{__builtin_amdgcn_readfirstlane((int)(u32)a), __builtin_amdgcn_readfirstlane((int)((u32)(a >> 32) & 0xffffu)), (int)bytes, 0x00020000};
}
__device__ __forceinline__ void dma16(i32x4 srd, u32 lds_addr, int voff, int soff) {
  u32 keep;
  asm volatile("s_mov_b32 %0, m0\n\ts_mov_b32 m0, %1\n\ts_nop 0\n\tbuffer_load_dwordx4 %2, %3, %4 offen lds\n\ts_mov_b32 m0, %0"
               : "=&s"(keep) : "s"(lds_addr), "v"(voff), "s"(srd), "s"(soff) : "memory");
}
template <int N> __device__ __forceinline__ void wait_vm() { asm volatile("s_waitcnt vmcnt(%0)" :: "i"(N) : "memory"); }

template <int I> struct IC { static constexpr int value = I; };
template <int N, int I = 0, class F> __device__ __forceinline__ void static_for(F&& f) {
  if constexpr (I < N) { f(IC<I>{}); static_for<N, I + 1>(f); }
}

#define LEAD 1
__device__ __forceinline__ int stage_of(int tau, int c) {
  if (tau < LEAD) return c + 4 * tau;
  const int u = tau - LEAD;
  constexpr int CUT = NSTEP - 4 * LEAD;
  if (u < CUT) return ((u & 3) == c) ? u + 4 * LEAD : u;
  const int v = u - CUT;
  const int blk = v / 3, r = v % 3;
  return CUT + 4 * blk + (r < c ? r : r + 1);
}
#define SB0 __builtin_amdgcn_sched_barrier(0)
__device__ __forceinline__ void bar() { __builtin_amdgcn_s_barrier(); asm volatile("" ::: "memory"); }

template <int MT> struct TileCfg {
  static constexpr int TM = 16 * MT;
  static constexpr int XI = (TM / 8 + 3) / 4;
  static constexpr int WB = BK * 128;
  static constexpr int XB = XI * 4 * 1024;
  static constexpr int SB = WB + XB;
  static constexpr int PER = 4 + XI;
};

template <int MT, int NSTG>
__device__ __forceinline__ void tile_loader(const Args& a, u32 sm0, int row0, int col0, int cls, int lw, int lane) {
  using C = TileCfg<MT>;
  const i32x4 rW = make_srd(a.w, (u32)KD * ND);
  const i32x4 rX = make_srd(a.x, (u32)MS * KD);
  const int wsig = ((lane >> 4) & 3) | ((lw & 1) << 2);
  const int wi0 = 8 * (lw >> 1) + 2 * (lw & 1);
  const int w_voff = (wi0 * 8 + (lane >> 3)) * ND + col0 + 16 * ((lane & 7) ^ wsig);
  const int xsig = (4 * (lw & 1) + (lane >> 4)) & 7;
  int x_voff[C::XI];
#pragma unroll
  for (int j = 0; j < C::XI; ++j) {
    const int i = lw + 4 * j;
    x_voff[j] = (i * 8 < C::TM) ? (row0 + 8 * i + (lane >> 3)) * KD + 16 * ((lane & 7) ^ xsig) : (int)0x7fffff00;
  }
  auto stage = [&](int tau, int slot) {
    const int tt = stage_of(tau, cls);
    const u32 base = sm0 + slot * C::SB;
#pragma unroll
    for (int j = 0; j < C::XI; ++j) dma16(rX, base + C::WB + (lw + 4 * j) * 1024, x_voff[j], tt * BK);
#pragma unroll
    for (int q = 0; q < 4; ++q) {
      const int di = 4 * (q >> 1) + (q & 1);
      dma16(rW, base + (wi0 + di) * 1024, w_voff, (tt * BK + di * 8) * ND);
    }
  };
#pragma unroll
  for (int t = 0; t < NSTG; ++t) stage(t, t);
  wait_vm<(NSTG - 1) * C::PER>();
  bar();
  wait_vm<(NSTG - 2) * C::PER>();
  bar();
  int slot = 0;
#pragma unroll 1
  for (int t = 1; t <= NSTEP - NSTG; ++t) {
    stage(t - 1 + NSTG, slot);
    slot = slot + 1 == NSTG ? 0 : slot + 1;
    wait_vm<(NSTG - 2) * C::PER>();
    bar();
  }
  static_for<NSTG - 2>([&](auto J) {
    wait_vm<(NSTG - 3 - J.value) * C::PER>();
    bar();
  });
}

template <int MT, int NSTG>
__device__ __forceinline__ void tile_consumer(lds_cp smem, int cw, int lane, i32x4 (&acc)[MT][2]) {
  using C = TileCfg<MT>;
  const int g = lane >> 4, q = (lane & 15) >> 1, p = lane & 1;
  const int rsig = (q >> 1) | ((g & 1) << 2);
  int w_rd[2];
#pragma unroll
  for (int nt = 0; nt < 2; ++nt) w_rd[nt] = (16 * g + q) * 128 + 16 * ((cw * 2 + nt) ^ rsig) + 8 * p;
  const int m = lane & 15;
  int x_rd[2];
#pragma unroll
  for (int s = 0; s < 2; ++s) x_rd[s] = C::WB + m * 128 + 16 * ((4 * s + g) ^ ((m >> 1) & 7));
#pragma unroll
  for (int i = 0; i < MT; ++i) { acc[i][0] = i32x4{0, 0, 0, 0}; acc[i][1] = i32x4{0, 0, 0, 0}; }
  struct Frag { i32x4 w[2]; i32x4 x[MT]; };
  auto rdw = [&](Frag& f, lds_cp base, int s, int nt) {
    i32x2 lo = __builtin_amdgcn_ds_read_tr8_b64_v2i32((lds_i2p)(base + w_rd[nt] + s * 64 * 128));
    i32x2 hi = __builtin_amdgcn_ds_read_tr8_b64_v2i32((lds_i2p)(base + w_rd[nt] + s * 64 * 128 + 8 * 128));
    f.w[nt] = i32x4{lo[0], lo[1], hi[0], hi[1]};
  };
  auto rd = [&](Frag& f, lds_cp base, int s) {
    rdw(f, base, s, 0);
    f.x[0] = *(lds_i4p)(base + x_rd[s]);
    rdw(f, base, s, 1);
#pragma unroll
    for (int i = 1; i < MT; ++i) f.x[i] = *(lds_i4p)(base + x_rd[s] + i * 16 * 128);
  };
  auto mma = [&](const Frag& f) {
#pragma unroll
    for (int i = 0; i < MT; ++i)
#pragma unroll
      for (int nt = 0; nt < 2; ++nt)
        acc[i][nt] = __builtin_amdgcn_mfma_i32_16x16x64_i8(f.w[nt], f.x[i], acc[i][nt], 0, 0, 0);
  };
#define ILV 1
  auto interleave = [&]() {
    __builtin_amdgcn_sched_group_barrier(0x100, 2, 0); __builtin_amdgcn_sched_group_barrier(0x8, 1, 0);
    __builtin_amdgcn_sched_group_barrier(0x100, 1, 0); __builtin_amdgcn_sched_group_barrier(0x8, 1, 0);
    __builtin_amdgcn_sched_group_barrier(0x100, 2, 0); __builtin_amdgcn_sched_group_barrier(0x8, 1, 0);
#pragma unroll
    for (int i = 1; i < MT; ++i) { __builtin_amdgcn_sched_group_barrier(0x100, 1, 0); __builtin_amdgcn_sched_group_barrier(0x8, 1, 0); }
    __builtin_amdgcn_sched_group_barrier(0x8, 2 * MT, 0);
  };
#define STSTAMP()
  Frag fa = {}, fb = {};
  bar(); SB0;
  STSTAMP();
  rd(fa, smem, 0); SB0;
  rd(fb, smem, 1); mma(fa); interleave(); SB0;
  int slot = 0;
#pragma unroll 1
  for (int t = 1; t < NSTEP; ++t) {
    slot = slot + 1 == NSTG ? 0 : slot + 1;
    lds_cp base = smem + slot * C::SB;
    asm volatile("s_waitcnt lgkmcnt(0)" ::: "memory");
    bar(); SB0;
    STSTAMP();
    rd(fa, base, 0); mma(fb); interleave(); SB0;
    rd(fb, base, 1); mma(fa); interleave(); SB0;
  }
  mma(fb);
}

__device__ __forceinline__ float epi(int acc, float s) { float y = (float)acc * s; return (y <= 0.f) ? 0.f : y; }

constexpr int PKT_REGS = 16;
constexpr int PKT_HDR = 128;
constexpr int PKT_CAP = PKT_HDR + PKT_REGS * 64 * 4;
constexpr int PKT_PER_SRC = 32 * 8;
constexpr size_t WS_PKT_OFF = 1u << 20;
constexpr u32 PKT_REGION = 2u * NR * PKT_PER_SRC * PKT_CAP;
constexpr int WS_TMO = 0;
constexpr int WS_CALL = 32768;

typedef __attribute__((address_space(3))) u32* lds_up;
typedef __attribute__((address_space(3))) unsigned short* lds_hp;

__device__ __forceinline__ u32 mbcnt64(u64 m) { return __builtin_amdgcn_mbcnt_hi((u32)(m >> 32), __builtin_amdgcn_mbcnt_lo((u32)m, 0u)); }

__device__ __forceinline__ u32 pkt_compact(lds_cp pk, const i32x4 (&acc)[4], float scale, int lane) {
  u32 y[PKT_REGS];
  u32 m = 0;
#pragma unroll
  for (int r = 0; r < PKT_REGS; ++r) {
    const float f = (float)acc[r >> 2][r & 3] * scale;
    const bool nz = !(f <= 0.f);
    y[r] = __builtin_bit_cast(u32, f);
    m |= nz ? (1u << r) : 0u;
  }
  if (m == 0) { m = 1; y[0] = 0x80000000u; }
  *(lds_hp)(pk + lane * 2) = (unsigned short)m;
  u32 base = 0;
#pragma unroll
  for (int r = 0; r < PKT_REGS; ++r) {
    const bool nz = (m >> r) & 1;
    const u64 bal = __builtin_amdgcn_ballot_w64(nz);
    if (nz) *(lds_up)(pk + PKT_HDR + (base + mbcnt64(bal)) * 4) = y[r];
    base += __builtin_popcountll(bal);
  }
  return PKT_HDR + base * 4;
}

__device__ __forceinline__ void dma16_sys(i32x4 srd, u32 lds_addr, int voff, int soff) {
  u32 keep;
  asm volatile("s_mov_b32 %0, m0\n\ts_mov_b32 m0, %1\n\ts_nop 0\n\tbuffer_load_dwordx4 %2, %3, %4 offen sc0 sc1 lds\n\ts_mov_b32 m0, %0"
               : "=&s"(keep) : "s"(lds_addr), "v"(voff), "s"(srd), "s"(soff) : "memory");
}

__device__ __forceinline__ void pkt_receive(const Args& a, lds_cp pk, u32 poff, int orow, int ocol, u32 call, int lane) {
  char* preg = (char*)a.ws[a.rank] + WS_PKT_OFF;
  const i32x4 rP = make_srd(preg, PKT_REGION);
  const u32 pka = (u32)(size_t)pk;
  const u64 t0 = __builtin_amdgcn_s_memrealtime();
  u32* tmo = a.ws[a.rank] + WS_TMO;
  u32 v[PKT_REGS];
  u32 total = 0;
  bool ok = false;
  int nch = 3;
  u32 spins = 0;
  auto expired = [&]() {
    if ((++spins & 15) != 0) return false;
    if (__hip_atomic_load(tmo, __ATOMIC_RELAXED, __HIP_MEMORY_SCOPE_AGENT) == call || __builtin_amdgcn_s_memrealtime() - t0 > DEADLINE) {
      __hip_atomic_store(tmo, call, __ATOMIC_RELAXED, __HIP_MEMORY_SCOPE_AGENT);
      return true;
    }
    return false;
  };
  bool dead = false;
  for (;;) {
    const u32 h = __hip_atomic_load((u32*)(preg + poff), __ATOMIC_RELAXED, __HIP_MEMORY_SCOPE_SYSTEM);
    if (__builtin_amdgcn_readfirstlane(h) != 0) break;
    if (expired()) { dead = true; break; }
    __builtin_amdgcn_s_sleep(HDRSLEEP);
  }
  while (!dead) {
    dma16_sys(rP, pka, lane * 16, poff);
    dma16_sys(rP, pka + 1024, lane * 16, poff + 1024);
    dma16_sys(rP, pka + 2048, lane * 16, poff + 2048);
    if (nch > 3) {
      dma16_sys(rP, pka + 3072, lane * 16, poff + 3072);
      if (lane < 8) dma16_sys(rP, pka + 4096, lane * 16, poff + 4096);
    }
    wait_vm<0>();
    const u32 m = *(lds_hp)(pk + lane * 2);
    if (__builtin_amdgcn_ballot_w64(m == 0) == 0) {
      u32 base = 0;
      u32 bad = 0;
#pragma unroll
      for (int r = 0; r < PKT_REGS; ++r) {
        const bool nz = (m >> r) & 1;
        const u64 bal = __builtin_amdgcn_ballot_w64(nz);
        const u32 w = *(lds_up)(pk + PKT_HDR + (base + mbcnt64(bal)) * 4);
        v[r] = nz ? w : 0u;
        bad |= (nz && w == 0) ? 1u : 0u;
        base += __builtin_popcountll(bal);
      }
      total = PKT_HDR + base * 4;
      if (total > 3072 && nch == 3) { nch = 5; continue; }
      if (__builtin_amdgcn_ballot_w64(bad != 0) == 0) { ok = true; break; }
    }
    if (expired()) break;
    __builtin_amdgcn_s_sleep(2);
  }
  auto rO = __builtin_amdgcn_make_buffer_rsrc((void*)a.out[a.rank], 0, (u32)(NR * MS * NS * 4), 0x00020000);
#pragma unroll
  for (int t = 0; t < 4; ++t) {
    i32x4 o;
#pragma unroll
    for (int e = 0; e < 4; ++e) o[e] = ok ? (int)(v[t * 4 + e] & 0x7fffffffu) : 0x7fc00000;
    const int r = orow + (t >> 1) * 16 + (lane & 15), c = ocol + (t & 1) * 16 + (lane >> 4) * 4;
    __builtin_amdgcn_raw_buffer_store_b128(o, rO, (r * NS + c) * 4, 0, OUT_AUX);
  }
  if (ok) {
    auto rC = __builtin_amdgcn_make_buffer_rsrc((void*)preg, 0, PKT_REGION, 0x00020000);
    const i32x4 z = {0, 0, 0, 0};
    for (u32 off = lane * 16; off < total; off += 1024) __builtin_amdgcn_raw_buffer_store_b128(z, rC, poff + off, 0, 17);
  }
}

__global__ void __launch_bounds__(NTHR, 2) fused_kernel(Args a) {
  extern __shared__ __attribute__((aligned(16))) char smem_raw[];
  lds_cp smem = (lds_cp)smem_raw;
  const int lane = threadIdx.x & 63;
  const int wave = __builtin_amdgcn_readfirstlane(threadIdx.x >> 6);
  const int bid = blockIdx.x;
  const int xcd = bid & 7, idx = bid >> 3;
  const int doff = (idx >> 3) * 2 + (xcd >> 2);
  const int dest = (doff + a.rank) & 7;
  const int sq = xcd & 3;
  const int col0 = dest * NS + sq * 128;
  const int rb = idx & 7;
  const int row0 = rb * 64;
  u32* myws = a.ws[a.rank];
  constexpr int MT = 4;
  lds_up sctl = (lds_up)(smem + SMEM - 1024);
  const int cls = ((idx >> 3) - (idx & 7)) & 3;
  i32x4 acc[MT][2];
  float scale = 0.f;
  if (wave < 4) {
    tile_loader<MT, NSTAGES>(a, (u32)(size_t)smem, row0, col0, cls, wave, lane);
    scale = a.sx[0] * a.sw[0];
  } else {
    u32 prev = 0;
    if (wave == 4) prev = __hip_atomic_load(myws + WS_CALL + bid * 32, __ATOMIC_RELAXED, __HIP_MEMORY_SCOPE_AGENT);
    scale = a.sx[0] * a.sw[0];
    tile_consumer<MT, NSTAGES>(smem, wave - 4, lane, acc);
    if (wave == 4 && lane == 0) {
      sctl[0] = prev + 1;
      sctl[9] = 0;
      __hip_atomic_store(myws + WS_CALL + bid * 32, prev + 1, __ATOMIC_RELAXED, __HIP_MEMORY_SCOPE_AGENT);
    }
  }
  const int cw = wave & 3;
  lds_cp pk = smem + 32768 + wave * PKT_CAP;
  asm volatile("s_waitcnt lgkmcnt(0)" ::: "memory");
  bar();
  const u32 call = __builtin_amdgcn_readfirstlane(sctl[0]);
  const u32 par = call & 1;
  if (doff == 0) {
    if (wave >= 4) {
      auto rO = __builtin_amdgcn_make_buffer_rsrc((void*)a.out[a.rank], 0, (u32)(NR * MS * NS * 4), 0x00020000);
#pragma unroll
      for (int i = 0; i < MT; ++i)
#pragma unroll
        for (int nt = 0; nt < 2; ++nt) {
          const int r = a.rank * MS + row0 + i * 16 + (lane & 15);
          const int c = sq * 128 + cw * 32 + nt * 16 + (lane >> 4) * 4;
          f32x4 v = {epi(acc[i][nt][0], scale), epi(acc[i][nt][1], scale), epi(acc[i][nt][2], scale), epi(acc[i][nt][3], scale)};
          __builtin_amdgcn_raw_buffer_store_b128(__builtin_bit_cast(i32x4, v), rO, (r * NS + c) * 4, 0, OUT_AUX);
        }
    }
    const int u = (sq * 8 + rb) * 8 + wave;
    if (u < 224) {
      const int w = 7;
      const int so = 1 + u % 7, tile = u / 7;
      const int src = (a.rank - so) & 7;
      pkt_receive(a, pk, ((par * NR + src) * PKT_PER_SRC + tile * 8 + w) * PKT_CAP,
                  src * MS + (tile & 7) * 64 + (w >> 2) * 32, (tile >> 3) * 128 + (w & 3) * 32, call, lane);
    }
  } else {
    i32x4 pa[4];
    if (wave >= 4) {
#pragma unroll
      for (int t = 0; t < 4; ++t) *(lds_i4p)(smem + cw * 4096 + t * 1024 + lane * 16) = acc[2 + (t >> 1)][t & 1];
#pragma unroll
      for (int t = 0; t < 4; ++t) pa[t] = acc[t >> 1][t & 1];
    }
    asm volatile("s_waitcnt lgkmcnt(0)" ::: "memory");
    bar();
    if (wave < 4) {
#pragma unroll
      for (int t = 0; t < 4; ++t) pa[t] = *(lds_i4p)(smem + cw * 4096 + t * 1024 + lane * 16);
    }
    const int myw = wave ^ 4;
    lds_cp pkw = smem + 32768 + myw * PKT_CAP;
    const u32 tb = pkt_compact(pkw, pa, scale, lane);
    if (lane == 0) sctl[1 + myw] = tb;
    asm volatile("s_waitcnt lgkmcnt(0)" ::: "memory");
    bar();
    const int tile = sq * 8 + rb;
    if (wave == 0) {
      auto rD = __builtin_amdgcn_make_buffer_rsrc((void*)((char*)a.ws[dest] + WS_PKT_OFF), 0, PKT_REGION, 0x00020000);
      const u32 dbase = ((par * NR + a.rank) * PKT_PER_SRC + tile * 8) * PKT_CAP;
#pragma unroll 1
      for (int w = 0; w < 8; ++w) {
        const u32 tbw = __builtin_amdgcn_readfirstlane(sctl[1 + w]);
#pragma unroll 1
        for (u32 off0 = 0; off0 < tbw; off0 += 1024) {
          const u32 off = off0 + lane * 16;
          const i32x4 d = *(lds_i4p)(smem + 32768 + w * PKT_CAP + off);
          wait_vm<PACE>();
          __builtin_amdgcn_raw_buffer_store_b128(d, rD, off < tbw ? dbase + w * PKT_CAP + off : 0x7fffff00u, 0, 17);
        }
        if (lane == 0) *(volatile __attribute__((address_space(3))) u32*)(sctl + 9) = w + 1;
      }
    } else {
      const int w = wave - 1;
      const int src = (a.rank - doff) & 7;
      while (*(volatile __attribute__((address_space(3))) u32*)(sctl + 9) <= (u32)w) __builtin_amdgcn_s_sleep(4);
      pkt_receive(a, smem + 72 * 1024 + (wave - 1) * PKT_CAP, ((par * NR + src) * PKT_PER_SRC + tile * 8 + w) * PKT_CAP,
                  src * MS + (tile & 7) * 64 + (w >> 2) * 32, (tile >> 3) * 128 + (w & 3) * 32, call, lane);
    }
  }
}
}

extern "C" void kernel_launch(int rank, int nranks, ncclComm_t comm, hipStream_t stream,
                              void* const* d_in, const int* in_sizes, int n_in,
                              void* d_out, int out_size, const int* in_dtypes, int out_dtype,
                              void* const* d_out_all, void* const* d_ws_all, size_t ws_bytes) {
  if (nranks != NR) return;
  if (ws_bytes < WS_PKT_OFF + PKT_REGION) return;
  static bool init[NR] = {};
  if ((unsigned)rank >= (unsigned)NR) return;
  if (!init[rank]) {
    HIP_CHECK(hipFuncSetAttribute((const void*)fused_kernel, hipFuncAttributeMaxDynamicSharedMemorySize, SMEM));
    init[rank] = true;
  }
  Args a;
  a.x = (const char*)d_in[0]; a.w = (const char*)d_in[1]; a.sx = (const float*)d_in[2]; a.sw = (const float*)d_in[3];
  for (int p = 0; p < NR; ++p) { a.out[p] = (float*)d_out_all[p]; a.ws[p] = (u32*)d_ws_all[p]; }
  a.rank = rank;
  fused_kernel<<<NWG, NTHR, SMEM, stream>>>(a);
}
```

```cpp
#include <hip/hip_runtime.h>
#include <rccl/rccl.h>
#include <cstdio>

#define HIP_CHECK(expr) do { hipError_t _e = (expr); if (_e != hipSuccess) { \
    fprintf(stderr, "HIP error %s at %s:%d\n", hipGetErrorString(_e), __FILE__, __LINE__); \
    return; } } while (0)

namespace {
typedef unsigned u32;
typedef unsigned long long u64;
typedef int i32x4 __attribute__((ext_vector_type(4)));
typedef int i32x2 __attribute__((ext_vector_type(2)));
typedef float f32x4 __attribute__((ext_vector_type(4)));
#define LDS __attribute__((address_space(3)))
typedef LDS i32x2* lds_i2p;
typedef LDS i32x4* lds_i4p;
typedef LDS char* lds_cp;
typedef LDS u32* lds_up;
typedef volatile LDS u32* lds_vup;
typedef LDS unsigned short* lds_hp;

constexpr int NR = 8;
constexpr int MS = 512;
constexpr int KD = 4096;
constexpr int ND = 4096;
constexpr int NS = 512;
constexpr int BK = 128;
constexpr int NSTEP = KD / BK;
constexpr int NWG = 256;
constexpr int NTHR = 512;
constexpr int NSTG = 4;
#define TILES 1, 1, 2, 2, 2, 2, 3, 3
#define TILESB 1, 2, 2, 3, 2, 2, 2, 2
#define DELEGS 0, 0, 0, 0, 1, 1, 3, 3
#define LEADS 0, 0, 0, 0, 8, 8, 8, 8
#define PACE1 12
#define PACE2 6
#define PACE3 4
#define PACE4 6
#define KLATE 99
#define PACEL 8
#define RXL_AUX 17
#define HDRSLEEP 8
#define OUT_AUX 17
#define PMAX 3
struct Tab { int p[8], r0[8], del[8], lead[8]; int ndel; int di[32], dk[32]; u32 pk_p, pk_r0, pk_del, pk_lead; u64 pk_di, pk_dk; };
constexpr Tab make_tab(bool alt) {
  Tab t = {{TILES}, {}, {DELEGS}, {LEADS}, 0, {}, {}, 0, 0, 0, 0, 0, 0};
  constexpr int pb[8] = {TILESB};
  if (alt) for (int i = 0; i < 8; ++i) t.p[i] = pb[i];
  int r = 0;
  for (int i = 0; i < 8; ++i) { t.r0[i] = r; r += t.p[i]; }
  for (int d = 15; d >= 1; --d) for (int i = 0; i < 8; ++i) if (t.del[i] >= d) { t.di[t.ndel] = i; t.dk[t.ndel] = 4 * t.p[i] - d; ++t.ndel; }
  for (int i = 0; i < 8; ++i) { t.pk_p |= (u32)t.p[i] << (4 * i); t.pk_r0 |= (u32)t.r0[i] << (4 * i); t.pk_del |= (u32)t.del[i] << (4 * i); t.pk_lead |= (u32)t.lead[i] << (4 * i); }
  for (int q = 0; q < t.ndel; ++q) { t.pk_di |= (u64)t.di[q] << (4 * q); t.pk_dk |= (u64)t.dk[q] << (4 * q); }
  return t;
}
constexpr Tab TAB = make_tab(false), TABB = make_tab(true);
static_assert(TABB.r0[7] + TABB.p[7] == 16 && TABB.r0[4] * 8 <= 64, "");
static_assert(TAB.r0[7] + TAB.p[7] == 16, "tiles must cover 512 rows");
static_assert(TAB.ndel <= 16, "");
constexpr int TLINES = TAB.r0[4] * 8;
static_assert(TLINES <= 64, "");

constexpr int SMEM = NSTG * (BK * 128 + PMAX * 4096) + 1024;
constexpr u64 DEADLINE = 50000000ull;

struct Args {
  const char* x; const char* w; const float* sx; const float* sw;
  float* out[NR];
  u32* ws[NR];
  int rank;
};

constexpr int PKT_REGS = 16;
constexpr int PKT_HDR = 128;
constexpr int PKT_CAP = PKT_HDR + PKT_REGS * 64 * 4;
constexpr int PKT_PER_SRC = 256;
constexpr size_t WS_PKT_OFF = 1u << 20;
constexpr u32 PKT_REGION = 2u * NR * PKT_PER_SRC * PKT_CAP;
constexpr int WS_TMO = 0;
constexpr int WS_CALL = 32768;

__device__ __forceinline__ i32x4 make_srd(const void* p, u32 bytes) {
  const u64 a = (u64)p;
  return i32x4{__builtin_amdgcn_readfirstlane((int)(u32)a), __builtin_amdgcn_readfirstlane((int)((u32)(a >> 32) & 0xffffu)), (int)bytes, 0x00020000};
}
__device__ __forceinline__ void dma4(i32x4 srd, u32 lds_addr, int voff, int soff) {
  u32 keep;
  asm volatile("s_mov_b32 %0, m0\n\ts_mov_b32 m0, %1\n\ts_nop 0\n\tbuffer_load_dword %2, %3, %4 offen lds\n\ts_mov_b32 m0, %0"
               : "=&s"(keep) : "s"(lds_addr), "v"(voff), "s"(srd), "s"(soff) : "memory");
}
template <int N> __device__ __forceinline__ void wait_vm() { asm volatile("s_waitcnt vmcnt(%0)" :: "i"(N) : "memory"); }
__device__ __forceinline__ void wait_lgkm0() { asm volatile("s_waitcnt lgkmcnt(0)" ::: "memory"); }
template <int I> struct IC { static constexpr int value = I; };
template <int N, int I = 0, class F> __device__ __forceinline__ void static_for(F&& f) {
  if constexpr (I < N) { f(IC<I>{}); static_for<N, I + 1>(f); }
}
#define SB0 __builtin_amdgcn_sched_barrier(0)
__device__ __forceinline__ void bar() { __builtin_amdgcn_s_barrier(); asm volatile("" ::: "memory"); }

__device__ __forceinline__ int stage_of(int tau, int c, int lead) {
  const int u = tau - lead;
  const int cut = NSTEP - 4 * lead;
  const int mid = ((u & 3) == c) ? u + 4 * lead : u;
  const int v = u - cut;
  const int blk = (v * 43) >> 7, r = v - 3 * blk;
  const int tail = cut + 4 * blk + (r < c ? r : r + 1);
  return tau < lead ? c + 4 * tau : (u < cut ? mid : tail);
}

template <int P> struct TileCfg {
  static constexpr int MT = 2 * P;
  static constexpr int WB = BK * 128;
  static constexpr int SB = WB + P * 4096;
    static constexpr int PER = 4 + P;
};

template <int P>
__device__ __forceinline__ void dma_stage(i32x4 rX, i32x4 rW, u32 ldsx, u32 ldsw, const int (&xv)[4], const int (&wv)[4], int sx, int sw) {
  u32 keep;
#define DL(v, r, so, off) "buffer_load_dwordx4 " v ", " r ", " so " offen offset:" #off " lds\n\t"
  if constexpr (P == 1)
    asm volatile("s_mov_b32 %0, m0\n\ts_mov_b32 m0, %1\n\ts_nop 0\n\t" DL("%7", "%3", "%5", 0)
                 "s_mov_b32 m0, %2\n\ts_nop 0\n\t" DL("%11", "%4", "%6", 0) DL("%12", "%4", "%6", 1024) DL("%13", "%4", "%6", 2048) DL("%14", "%4", "%6", 3072) "s_mov_b32 m0, %0"
                 : "=&s"(keep) : "s"(ldsx), "s"(ldsw), "s"(rX), "s"(rW), "s"(sx), "s"(sw), "v"(xv[0]), "v"(xv[0]), "v"(xv[0]), "v"(xv[0]), "v"(wv[0]), "v"(wv[1]), "v"(wv[2]), "v"(wv[3]) : "memory");
  else if constexpr (P == 2)
    asm volatile("s_mov_b32 %0, m0\n\ts_mov_b32 m0, %1\n\ts_nop 0\n\t" DL("%7", "%3", "%5", 0) DL("%8", "%3", "%5", 1024)
                 "s_mov_b32 m0, %2\n\ts_nop 0\n\t" DL("%11", "%4", "%6", 0) DL("%12", "%4", "%6", 1024) DL("%13", "%4", "%6", 2048) DL("%14", "%4", "%6", 3072) "s_mov_b32 m0, %0"
                 : "=&s"(keep) : "s"(ldsx), "s"(ldsw), "s"(rX), "s"(rW), "s"(sx), "s"(sw), "v"(xv[0]), "v"(xv[1]), "v"(xv[0]), "v"(xv[0]), "v"(wv[0]), "v"(wv[1]), "v"(wv[2]), "v"(wv[3]) : "memory");
  else if constexpr (P == 3)
    asm volatile("s_mov_b32 %0, m0\n\ts_mov_b32 m0, %1\n\ts_nop 0\n\t" DL("%7", "%3", "%5", 0) DL("%8", "%3", "%5", 1024) DL("%9", "%3", "%5", 2048)
                 "s_mov_b32 m0, %2\n\ts_nop 0\n\t" DL("%11", "%4", "%6", 0) DL("%12", "%4", "%6", 1024) DL("%13", "%4", "%6", 2048) DL("%14", "%4", "%6", 3072) "s_mov_b32 m0, %0"
                 : "=&s"(keep) : "s"(ldsx), "s"(ldsw), "s"(rX), "s"(rW), "s"(sx), "s"(sw), "v"(xv[0]), "v"(xv[1]), "v"(xv[2]), "v"(xv[0]), "v"(wv[0]), "v"(wv[1]), "v"(wv[2]), "v"(wv[3]) : "memory");
  else
    asm volatile("s_mov_b32 %0, m0\n\ts_mov_b32 m0, %1\n\ts_nop 0\n\t" DL("%7", "%3", "%5", 0) DL("%8", "%3", "%5", 1024) DL("%9", "%3", "%5", 2048) DL("%10", "%3", "%5", 3072)
                 "s_mov_b32 m0, %2\n\ts_nop 0\n\t" DL("%11", "%4", "%6", 0) DL("%12", "%4", "%6", 1024) DL("%13", "%4", "%6", 2048) DL("%14", "%4", "%6", 3072) "s_mov_b32 m0, %0"
                 : "=&s"(keep) : "s"(ldsx), "s"(ldsw), "s"(rX), "s"(rW), "s"(sx), "s"(sw), "v"(xv[0]), "v"(xv[1]), "v"(xv[2]), "v"(xv[3]), "v"(wv[0]), "v"(wv[1]), "v"(wv[2]), "v"(wv[3]) : "memory");
}

template <int P>
__device__ __forceinline__ void tile_loader(const Args& a, u32 sm0, int row0, int col0, int cls, int lead, int trow, int tlines, int lw, int lane) {
  using C = TileCfg<P>;
  const i32x4 rW = make_srd(a.w, (u32)KD * ND);
  const i32x4 rX = make_srd(a.x, (u32)MS * KD);
  int wv[4], xv[4];
#pragma unroll
  for (int q = 0; q < 4; ++q) {
    const int wsig = ((lane >> 4) & 3) | ((q >> 1) << 2);
    wv[q] = ((4 * lw + q) * 8 + (lane >> 3)) * ND + col0 + 16 * ((lane & 7) ^ wsig) - q * 1024;
    const int i = lw * P + q;
    const int xsig = (4 * (i & 1) + (lane >> 4)) & 7;
    xv[q] = (row0 + 8 * i + (lane >> 3)) * KD + 16 * ((lane & 7) ^ xsig) - q * 1024;
  }
  const int order = stage_of(lane & 31, cls, lead);
  const u32 lw0 = sm0 + lw * 4096, lx0 = sm0 + C::WB + lw * P * 1024;
  auto stage = [&](int tt, int slot) { dma_stage<P>(rX, rW, lx0 + slot * C::SB, lw0 + slot * C::SB, xv, wv, tt * BK, tt * BK * ND); };
  const int t_voff = lane < tlines ? (trow + lane) * KD : (int)0x7fff0000;
  if (trow >= 0) dma4(rX, sm0 + SMEM - 512, t_voff, (cls + 4 * lw) * BK);
#pragma unroll
  for (int t = 0; t < NSTG; ++t) stage(__builtin_amdgcn_readfirstlane(stage_of(t, cls, lead)), t);
  if (trow >= 0) dma4(rX, sm0 + SMEM - 512, t_voff, (cls + 4 * lw + 16) * BK);
  int sx, sw; u32 ax, aw;
  auto prep = [&](int t) {
    const int tt = __builtin_amdgcn_readlane(order, (t - 1 + NSTG) & 31);
    const u32 so = ((t - 1) & (NSTG - 1)) * C::SB;
    sx = tt * BK; sw = tt * BK * ND; ax = lx0 + so; aw = lw0 + so;
    asm volatile("" : "+s"(sx), "+s"(sw), "+s"(ax), "+s"(aw));
  };
  prep(1);
  wait_vm<(NSTG - 1) * C::PER>();
  bar();
  wait_vm<(NSTG - 2) * C::PER>();
  bar();
#pragma unroll 1
  for (int t = 1; t <= NSTEP - NSTG; ++t) {
    dma_stage<P>(rX, rW, ax, aw, xv, wv, sx, sw);
    prep(t + 1);
    wait_vm<(NSTG - 2) * C::PER>();
    bar();
  }
  static_for<NSTG - 2>([&](auto J) {
    wait_vm<(NSTG - 3 - J.value) * C::PER>();
    bar();
  });
}

template <int P>
__device__ __forceinline__ void tile_consumer(lds_cp smem, int cw, int lane, i32x4 (&acc)[2 * P][2]) {
  using C = TileCfg<P>;
  constexpr int MT = C::MT;
  const int g = lane >> 4, q = (lane & 15) >> 1, p = lane & 1;
  const int rsig = (q >> 1) | ((g & 1) << 2);
  const int m = lane & 15;
  const u32 sm0 = (u32)(size_t)smem;
  u32 off[4];
  off[0] = sm0 + (16 * g + q) * 128 + 16 * ((cw * 2 + 0) ^ rsig) + 8 * p;
  off[1] = sm0 + (16 * g + q) * 128 + 16 * ((cw * 2 + 1) ^ rsig) + 8 * p;
  off[2] = sm0 + C::WB + m * 128 + 16 * ((0 + g) ^ ((m >> 1) & 7));
  off[3] = sm0 + C::WB + m * 128 + 16 * ((4 + g) ^ ((m >> 1) & 7));
#pragma unroll
  for (int i = 0; i < MT; ++i) { acc[i][0] = i32x4{0, 0, 0, 0}; acc[i][1] = i32x4{0, 0, 0, 0}; }
  struct Frag { i32x4 w[2]; i32x4 x[MT]; };
  auto rdw = [&](u32 ad, int s) {
    i32x2 lo = __builtin_amdgcn_ds_read_tr8_b64_v2i32((lds_i2p)(size_t)(ad + s * 64 * 128));
    i32x2 hi = __builtin_amdgcn_ds_read_tr8_b64_v2i32((lds_i2p)(size_t)(ad + s * 64 * 128 + 8 * 128));
    return i32x4{lo[0], lo[1], hi[0], hi[1]};
  };
  auto rdx = [&](u32 ad, int i) { return *(lds_i4p)(size_t)(ad + i * 16 * 128); };
  auto rdA = [&](Frag& f, const u32 (&ad)[4]) {
    f.w[0] = rdw(ad[0], 0); f.x[0] = rdx(ad[2], 0); f.w[1] = rdw(ad[1], 0);
#pragma unroll
    for (int i = 1; i < MT; ++i) f.x[i] = rdx(ad[2], i);
  };
  auto rdB = [&](Frag& f, const u32 (&ad)[4]) {
#pragma unroll
    for (int i = 0; i < MT - 2; ++i) f.x[i] = rdx(ad[3], i);
    f.w[0] = rdw(ad[0], 1); f.w[1] = rdw(ad[1], 1);
    f.x[MT - 2] = rdx(ad[3], MT - 2); f.x[MT - 1] = rdx(ad[3], MT - 1);
  };
  auto mma = [&](const Frag& f) {
#pragma unroll
    for (int i = 0; i < MT; ++i)
#pragma unroll
      for (int nt = 0; nt < 2; ++nt)
        acc[i][nt] = __builtin_amdgcn_mfma_i32_16x16x64_i8(f.w[nt], f.x[i], acc[i][nt], 0, 0, 0);
  };
#define SGB(m, n) __builtin_amdgcn_sched_group_barrier(m, n, 0)
  auto interleave = [&]() {
    SGB(0x8, 1); SGB(0x100, 2); SGB(0x8, 1); SGB(0x100, 1); SGB(0x8, 1); SGB(0x100, 2);
#pragma unroll
    for (int i = 1; i < MT; ++i) { SGB(0x8, 1); SGB(0x100, 1); }
#pragma unroll
    for (int i = 0; i < MT - 2; ++i) { SGB(0x8, 1); SGB(0x100, 1); }
    SGB(0x8, 1); SGB(0x100, 2); SGB(0x8, 1); SGB(0x100, 2);
    SGB(0x8, 1); SGB(0x100, 1); SGB(0x8, 1); SGB(0x100, 1);
    SGB(0x8, 4 * MT);
  };
  Frag fa, fb;
  u32 ad[4];
#pragma unroll
  for (int k = 0; k < 4; ++k) ad[k] = off[k];
  bar(); SB0;
  rdA(fa, ad); rdB(fb, ad); mma(fa); SB0;
  Frag fc;
  auto step = [&](int t, const Frag& fo, Frag& fn) {
    const u32 sb = (t & (NSTG - 1)) * C::SB;
#pragma unroll
    for (int k = 0; k < 4; ++k) { ad[k] = off[k] + sb; asm volatile("" : "+v"(ad[k])); }
    SB0;
    wait_lgkm0();
    bar(); SB0;
    rdA(fa, ad); rdB(fn, ad); mma(fo); mma(fa); interleave();
    SB0;
  };
#pragma unroll 1
  for (int t = 1; t < NSTEP - 1; t += 2) { step(t, fb, fc); step(t + 1, fc, fb); }
  step(NSTEP - 1, fb, fc);
  mma(fc);
}

__device__ __forceinline__ float epi(int acc, float s) { float y = (float)acc * s; return (y <= 0.f) ? 0.f : y; }
constexpr int PKT_LDS = PKT_CAP + 16;

__device__ __forceinline__ u32 wave_scan(u32 x) {
  x += __builtin_amdgcn_update_dpp(0u, x, 0x111, 0xf, 0xf, false);
  x += __builtin_amdgcn_update_dpp(0u, x, 0x112, 0xf, 0xf, false);
  x += __builtin_amdgcn_update_dpp(0u, x, 0x114, 0xf, 0xf, false);
  x += __builtin_amdgcn_update_dpp(0u, x, 0x118, 0xf, 0xf, false);
  x += __builtin_amdgcn_update_dpp(0u, x, 0x142, 0xa, 0xf, false);
  x += __builtin_amdgcn_update_dpp(0u, x, 0x143, 0xc, 0xf, false);
  return x;
}

__device__ __forceinline__ u32 pkt_compact(u32 pk, const i32x4& a0, const i32x4& a1, const i32x4& a2, const i32x4& a3, float scale, int lane) {
  float y[PKT_REGS];
#pragma unroll
  for (int r = 0; r < PKT_REGS; ++r) {
    const int av = (r >> 2) == 0 ? a0[r & 3] : (r >> 2) == 1 ? a1[r & 3] : (r >> 2) == 2 ? a2[r & 3] : a3[r & 3];
    y[r] = (float)av * scale;
  }
  u64 k[PKT_REGS];
  u32 m = 0;
#define CMP8(o) asm volatile( \
    "v_cmp_nge_f32 %0, 0, %9\n\tv_cmp_nge_f32 %1, 0, %10\n\tv_cmp_nge_f32 %2, 0, %11\n\tv_cmp_nge_f32 %3, 0, %12\n\t" \
    "v_cmp_nge_f32 %4, 0, %13\n\tv_cmp_nge_f32 %5, 0, %14\n\tv_cmp_nge_f32 %6, 0, %15\n\tv_cmp_nge_f32 %7, 0, %16\n\t" \
    "v_addc_co_u32 %8, vcc, %8, %8, %0\n\tv_addc_co_u32 %8, vcc, %8, %8, %1\n\tv_addc_co_u32 %8, vcc, %8, %8, %2\n\tv_addc_co_u32 %8, vcc, %8, %8, %3\n\t" \
    "v_addc_co_u32 %8, vcc, %8, %8, %4\n\tv_addc_co_u32 %8, vcc, %8, %8, %5\n\tv_addc_co_u32 %8, vcc, %8, %8, %6\n\tv_addc_co_u32 %8, vcc, %8, %8, %7\n\ts_nop 1" \
    : "=&s"(k[o]), "=&s"(k[o + 1]), "=&s"(k[o + 2]), "=&s"(k[o + 3]), "=&s"(k[o + 4]), "=&s"(k[o + 5]), "=&s"(k[o + 6]), "=&s"(k[o + 7]), "+v"(m) \
    : "v"(y[o]), "v"(y[o + 1]), "v"(y[o + 2]), "v"(y[o + 3]), "v"(y[o + 4]), "v"(y[o + 5]), "v"(y[o + 6]), "v"(y[o + 7]) : "vcc")
  CMP8(0); CMP8(8);
  const bool z = m == 0;
  y[0] = z ? -0.0f : y[0];
  m = z ? 0x8000u : m;
  k[0] = __builtin_amdgcn_ballot_w64(m >= 0x8000u);
  const u32 c = __builtin_popcount(m);
  const u32 inc = wave_scan(c);
  const u32 n = __builtin_amdgcn_readlane(inc, 63);
  *(lds_hp)(size_t)(pk + lane * 2) = (unsigned short)m;
  if (lane < 3) *(lds_up)(size_t)(pk + PKT_HDR + (n + lane) * 4) = 0x80000000u;
  u32 ad = pk + PKT_HDR + (inc - c) * 4;
#define PUT1(i, j) "s_mov_b64 exec, %" #i "\n\tds_write_b32 %0, %" #j "\n\tv_add_u32 %0, 4, %0\n\t"
#define PUT8(o) asm volatile(PUT1(1, 9) PUT1(2, 10) PUT1(3, 11) PUT1(4, 12) PUT1(5, 13) PUT1(6, 14) PUT1(7, 15) PUT1(8, 16) "s_mov_b64 exec, -1" \
    : "+v"(ad) : "s"(k[o]), "s"(k[o + 1]), "s"(k[o + 2]), "s"(k[o + 3]), "s"(k[o + 4]), "s"(k[o + 5]), "s"(k[o + 6]), "s"(k[o + 7]), \
      "v"(y[o]), "v"(y[o + 1]), "v"(y[o + 2]), "v"(y[o + 3]), "v"(y[o + 4]), "v"(y[o + 5]), "v"(y[o + 6]), "v"(y[o + 7]) : "memory")
  PUT8(0); PUT8(8);
  return (PKT_HDR + n * 4 + 15) & ~15u;
}

__device__ __forceinline__ bool has_zero(const i32x4& d) {
  u32 mn = min(min((u32)d[0], (u32)d[1]), min((u32)d[2], (u32)d[3]));
  return mn == 0;
}

__device__ __forceinline__ void pkt_receive(const Args& a, u32 pk, u32 poff, int orow, int ocol, u32 call, int lane, bool tail) {
  char* preg = (char*)a.ws[a.rank] + WS_PKT_OFF;
  auto rP = __builtin_amdgcn_make_buffer_rsrc((void*)preg, 0, PKT_REGION, 0x00020000);
  const u64 t0 = __builtin_amdgcn_s_memrealtime();
  u32* tmo = a.ws[a.rank] + WS_TMO;
  u32 spins = 0;
  auto expired = [&]() {
    if ((++spins & 15) != 0) return false;
    if (__hip_atomic_load(tmo, __ATOMIC_RELAXED, __HIP_MEMORY_SCOPE_AGENT) == call || __builtin_amdgcn_s_memrealtime() - t0 > DEADLINE) {
      __hip_atomic_store(tmo, call, __ATOMIC_RELAXED, __HIP_MEMORY_SCOPE_AGENT);
      return true;
    }
    return false;
  };
  bool ok = true;
  u32 m, c, inc, total;
  const u32 lo = lane * 16;
  i32x4 d0, d1, d2;
  {
    for (;;) {
      m = __builtin_amdgcn_raw_buffer_load_b16(rP, lane * 2, poff, 17);
      if (__builtin_amdgcn_ballot_w64(m == 0) == 0) break;
      if (expired()) { ok = false; break; }
      __builtin_amdgcn_s_sleep(HDRSLEEP);
      asm volatile("" ::: "memory");
    }
    d0 = __builtin_amdgcn_raw_buffer_load_b128(rP, lo, poff, 17);
    d1 = __builtin_amdgcn_raw_buffer_load_b128(rP, lo + 1024, poff, 17);
    d2 = __builtin_amdgcn_raw_buffer_load_b128(rP, lo + 2048, poff, 17);
  }
  c = __builtin_popcount(m);
  inc = wave_scan(c);
  total = (PKT_HDR + __builtin_amdgcn_readlane(inc, 63) * 4 + 15) & ~15u;
  u64 k[PKT_REGS];
  u32 ms = m << 16;
#define MSK1(i) "v_add_co_u32 %8, %" #i ", %8, %8\n\t"
#define MSK8(o) asm volatile(MSK1(0) MSK1(1) MSK1(2) MSK1(3) MSK1(4) MSK1(5) MSK1(6) MSK1(7) "s_nop 1" \
    : "=&s"(k[o]), "=&s"(k[o + 1]), "=&s"(k[o + 2]), "=&s"(k[o + 3]), "=&s"(k[o + 4]), "=&s"(k[o + 5]), "=&s"(k[o + 6]), "=&s"(k[o + 7]), "+v"(ms))
  MSK8(0); MSK8(8);
  while (ok) {
    const bool bad = ((lo < total) & has_zero(d0)) | ((lo + 1024 < total) & has_zero(d1)) | ((lo + 2048 < total) & has_zero(d2));
    if (__builtin_amdgcn_ballot_w64(bad) == 0) break;
    if (expired()) { ok = false; break; }
    __builtin_amdgcn_s_sleep(1);
    asm volatile("" ::: "memory");
    d0 = __builtin_amdgcn_raw_buffer_load_b128(rP, lo, poff, 17);
    d1 = __builtin_amdgcn_raw_buffer_load_b128(rP, lo + 1024, poff, 17);
    d2 = __builtin_amdgcn_raw_buffer_load_b128(rP, lo + 2048, poff, 17);
  }
  *(lds_i4p)(size_t)(pk + lo) = d0;
  *(lds_i4p)(size_t)(pk + lo + 1024) = d1;
  *(lds_i4p)(size_t)(pk + lo + 2048) = d2;
  if (total > 3072) {
    const u32 o4 = lane < 8 ? lo + 4096 : 0x7fffff00u;
    while (ok) {
      d0 = __builtin_amdgcn_raw_buffer_load_b128(rP, lo + 3072, poff, 17);
      d1 = __builtin_amdgcn_raw_buffer_load_b128(rP, o4, poff, 17);
      const bool bad = ((lo + 3072 < total) & has_zero(d0)) | ((lo + 4096 < total) & has_zero(d1));
      if (__builtin_amdgcn_ballot_w64(bad) == 0) break;
      if (expired()) { ok = false; break; }
      __builtin_amdgcn_s_sleep(1);
      asm volatile("" ::: "memory");
    }
    *(lds_i4p)(size_t)(pk + lo + 3072) = d0;
    if (lane < 8) *(lds_i4p)(size_t)(pk + lo + 4096) = d1;
  }
  u32 v[PKT_REGS];
#pragma unroll
  for (int r = 0; r < PKT_REGS; ++r) v[r] = 0;
  u32 ad = pk + PKT_HDR + (inc - c) * 4;
#define GET1(i, j) "s_mov_b64 exec, %" #i "\n\tds_read_b32 %" #j ", %0\n\tv_add_u32 %0, 4, %0\n\t"
#define GET8(o, tail) asm volatile(GET1(9, 1) GET1(10, 2) GET1(11, 3) GET1(12, 4) GET1(13, 5) GET1(14, 6) GET1(15, 7) GET1(16, 8) "s_mov_b64 exec, -1" tail \
    : "+v"(ad), "+v"(v[o]), "+v"(v[o + 1]), "+v"(v[o + 2]), "+v"(v[o + 3]), "+v"(v[o + 4]), "+v"(v[o + 5]), "+v"(v[o + 6]), "+v"(v[o + 7]) \
    : "s"(k[o]), "s"(k[o + 1]), "s"(k[o + 2]), "s"(k[o + 3]), "s"(k[o + 4]), "s"(k[o + 5]), "s"(k[o + 6]), "s"(k[o + 7]) : "memory")
  GET8(0, ""); GET8(8, "");
  asm volatile("s_waitcnt lgkmcnt(0)" : "+v"(v[0]), "+v"(v[1]), "+v"(v[2]), "+v"(v[3]), "+v"(v[4]), "+v"(v[5]), "+v"(v[6]), "+v"(v[7]),
               "+v"(v[8]), "+v"(v[9]), "+v"(v[10]), "+v"(v[11]), "+v"(v[12]), "+v"(v[13]), "+v"(v[14]), "+v"(v[15]) :: "memory");
  v[0] &= 0x7fffffffu;
  auto rO = __builtin_amdgcn_make_buffer_rsrc((void*)a.out[a.rank], 0, (u32)(NR * MS * NS * 4), 0x00020000);
  if (!ok) {
#pragma unroll
    for (int r = 0; r < PKT_REGS; ++r) v[r] = 0x7fc00000u;
  }
#pragma unroll
  for (int t = 0; t < 4; ++t) {
    const i32x4 o = {(int)v[t * 4], (int)v[t * 4 + 1], (int)v[t * 4 + 2], (int)v[t * 4 + 3]};
    const int r = orow + (t >> 1) * 16 + (lane & 15), cc = ocol + (t & 1) * 16 + (lane >> 4) * 4;
    if (RXL_AUX != OUT_AUX && tail) __builtin_amdgcn_raw_buffer_store_b128(o, rO, (r * NS + cc) * 4, 0, RXL_AUX);
    else __builtin_amdgcn_raw_buffer_store_b128(o, rO, (r * NS + cc) * 4, 0, OUT_AUX);
  }
  if (ok) {
    const i32x4 z = {0, 0, 0, 0};
    if (RXL_AUX != 17 && tail) { for (u32 off = lo; off < total; off += 1024) __builtin_amdgcn_raw_buffer_store_b128(z, rP, poff + off, 0, RXL_AUX); }
    else for (u32 off = lo; off < total; off += 1024) __builtin_amdgcn_raw_buffer_store_b128(z, rP, poff + off, 0, 17);
  }
}

template <int P>
__device__ __forceinline__ void tile_gemm(const Args& a, lds_cp smem, lds_vup sctl, int row0, int col0, int cls, int lead, int trow, int tlines, bool local, int wave, int lane, u64& tsa, u64& tsb) {
  if (wave < 4) {
    tile_loader<P>(a, (u32)(size_t)smem, row0, col0, cls, lead, trow, tlines, wave, lane);
  } else {
    const int cw = wave - 4;
    const float scale = a.sx[0] * a.sw[0];
    i32x4 acc[2 * P][2];
    tile_consumer<P>(smem, cw, lane, acc);
    if (local) {
      auto rO = __builtin_amdgcn_make_buffer_rsrc((void*)a.out[a.rank], 0, (u32)(NR * MS * NS * 4), 0x00020000);
#pragma unroll
      for (int i = 0; i < 2 * P; ++i)
#pragma unroll
        for (int nt = 0; nt < 2; ++nt) {
          const int r = a.rank * MS + row0 + i * 16 + (lane & 15);
          const int c = (col0 & (NS - 1)) + cw * 32 + nt * 16 + (lane >> 4) * 4;
          f32x4 v = {epi(acc[i][nt][0], scale), epi(acc[i][nt][1], scale), epi(acc[i][nt][2], scale), epi(acc[i][nt][3], scale)};
          __builtin_amdgcn_raw_buffer_store_b128(__builtin_bit_cast(i32x4, v), rO, (r * NS + c) * 4, 0, OUT_AUX);
        }
    } else {
#pragma unroll
      for (int pI = 0; pI < P; ++pI) {
        const int k = pI * 4 + cw;
        const u32 tb = pkt_compact((u32)(size_t)smem + k * PKT_LDS, acc[2 * pI][0], acc[2 * pI][1], acc[2 * pI + 1][0], acc[2 * pI + 1][1], scale, lane);
        wait_lgkm0();
        if (lane == 0) sctl[1 + k] = tb;
      }
    }
  }
}

__global__ void __launch_bounds__(NTHR, 2) fused_kernel(Args a) {
  extern __shared__ __attribute__((aligned(16))) char smem_raw[];
  lds_cp smem = (lds_cp)smem_raw;
  const int lane = threadIdx.x & 63;
  const int wave = __builtin_amdgcn_readfirstlane(threadIdx.x >> 6);
  const int bid = blockIdx.x;
  const int xcd = bid & 7, idx = bid >> 3;
  const int tj = idx & 3, ti = idx >> 2;
  const int doff = tj * 2 + (xcd >> 2);
  const int dest = (doff + a.rank) & 7;
  const int sq = xcd & 3;
  const int col0 = dest * NS + sq * 128;
  const bool slow = ((((u32)(size_t)a.w + col0) >> 7) & 7) == 3;
  const int PA = (TAB.pk_p >> (4 * ti)) & 15, r32A = (TAB.pk_r0 >> (4 * ti)) & 15;
  const int P = slow ? (TABB.pk_p >> (4 * ti)) & 15 : PA, r32 = slow ? (TABB.pk_r0 >> (4 * ti)) & 15 : r32A;
  const int tlines = slow ? TABB.r0[4] * 8 : TAB.r0[4] * 8;
  const int lead = (TAB.pk_lead >> (4 * ti)) & 15;
  const int cls = (tj - ti) & 3;
  u32* myws = a.ws[a.rank];
  lds_vup sctl = (lds_vup)(smem + SMEM - 1024);
  int cpf = 0;
  if (wave == 7) {
    u64 pc; int clen;
    asm volatile("s_getpc_b64 %0\nfk_code_a:\n\ts_mov_b32 %1, fk_code_b-fk_code_a" : "=s"(pc), "=s"(clen));
    const int co = (idx * 8 + (lane & 7)) * 128;
    const char* cq = (const char*)pc + (co < clen ? co : 0);
    asm volatile("global_load_dword %0, %1, off" : "=v"(cpf) : "v"(cq) : "memory");
  }
  if (wave == 4) {
    const u32 prev = __hip_atomic_load(myws + WS_CALL + bid * 32, __ATOMIC_RELAXED, __HIP_MEMORY_SCOPE_AGENT);
    if (lane < 32) sctl[lane] = lane == 0 ? prev + 1 : 0u;
    wait_lgkm0();
  }
  const bool local = doff == 0;
  const int trow = ti >= 4 ? (ti - 4) * tlines : -1;
  u64 tsa = 0, tsb = 0;
  if (P == 1) tile_gemm<1>(a, smem, sctl, r32 * 32, col0, cls, lead, trow, tlines, local, wave, lane, tsa, tsb);
  else if (P == 2) tile_gemm<2>(a, smem, sctl, r32 * 32, col0, cls, lead, trow, tlines, local, wave, lane, tsa, tsb);
  else tile_gemm<3>(a, smem, sctl, r32 * 32, col0, cls, lead, trow, tlines, local, wave, lane, tsa, tsb);
  asm volatile("s_waitcnt vmcnt(0)" :: "v"(cpf) : "memory");
  const u32 call = __builtin_amdgcn_readfirstlane(sctl[0]);
  const u32 par = call & 1;
  if (wave == 4 && lane == 0) __hip_atomic_store(myws + WS_CALL + bid * 32, call, __ATOMIC_RELAXED, __HIP_MEMORY_SCOPE_AGENT);
  if (!local && wave == 0) {
    auto rD = __builtin_amdgcn_make_buffer_rsrc((void*)((char*)a.ws[dest] + WS_PKT_OFF), 0, PKT_REGION, 0x00020000);
    auto send = [&](auto PC) {
    const u32 lo = lane * 16;
    u32 tbs = 0;
#pragma unroll 1
    for (int k = 0; k < 4 * P; ++k) {
      u32 tb;
      while ((tb = __builtin_amdgcn_readlane(tbs, k)) == 0) tbs = sctl[1 + (lane & 15)];
      asm volatile("" ::: "memory");
      lds_cp src = smem + k * PKT_LDS + lo;
      const i32x4 d0 = *(lds_i4p)(src), d1 = *(lds_i4p)(src + 1024), d2 = *(lds_i4p)(src + 2048);
      const u32 dst = ((par * NR + a.rank) * PKT_PER_SRC + (r32 + (k >> 2)) * 16 + sq * 4 + (k & 3)) * PKT_CAP + lo;
      wait_vm<PC.value>();
      __builtin_amdgcn_raw_buffer_store_b128(d0, rD, lo < tb ? dst : 0x7fff0000u, 0, 17);
      if (tb > 1024) {
        wait_vm<PC.value>();
        __builtin_amdgcn_raw_buffer_store_b128(d1, rD, lo + 1024 < tb ? dst + 1024 : 0x7fff0000u, 0, 17);
      }
      if (tb > 2048) {
        wait_vm<PC.value>();
        __builtin_amdgcn_raw_buffer_store_b128(d2, rD, lo + 2048 < tb ? dst + 2048 : 0x7fff0000u, 0, 17);
      }
#pragma unroll 1
      for (u32 off0 = 3072; off0 < tb; off0 += 1024) {
        const i32x4 d = *(lds_i4p)(src + off0);
        wait_vm<PC.value>();
        __builtin_amdgcn_raw_buffer_store_b128(d, rD, lo + off0 < tb ? dst + off0 : 0x7fff0000u, 0, 17);
      }
    }
    };
    if (ti < 2) send(IC<PACE1>{}); else if (ti < 4) send(IC<PACE2>{}); else if (ti < 6) send(IC<PACE3>{}); else send(IC<PACE4>{});
  } else {
    const u32 rxb = (u32)(size_t)smem + (local ? wave : 4 * P + wave - 1) * PKT_LDS;
    const int nq = local ? TAB.ndel * 28 : 4 * PA - (int)((TAB.pk_del >> (4 * ti)) & 15);
    const int q0 = local ? (sq * 8 + ti) * 8 + wave : wave - 1;
    const int qs = local ? 256 : 7;
#pragma unroll 1
    for (int q = q0; q < nq; q += qs) {
      int so, strip, rr, k;
      if (local) {
        const int pi = q / 28, rest = q % 28;
        so = 1 + rest % 7; strip = rest / 7;
        const int i2 = (int)(TAB.pk_di >> (4 * pi)) & 15;
        k = (int)(TAB.pk_dk >> (4 * pi)) & 15;
        rr = (TAB.pk_r0 >> (4 * i2)) & 15;
      } else { so = doff; strip = sq; rr = r32A; k = q; }
      const int src = (a.rank - so) & 7;
      const int rb = rr + (k >> 2), cg = strip * 4 + (k & 3);
      pkt_receive(a, rxb, ((par * NR + src) * PKT_PER_SRC + rb * 16 + cg) * PKT_CAP, src * MS + rb * 32, cg * 32, call, lane, local);
    }
  }
  asm volatile("fk_code_b:" ::: "memory");
}
}

extern "C" void kernel_launch(int rank, int nranks, ncclComm_t comm, hipStream_t stream,
                              void* const* d_in, const int* in_sizes, int n_in,
                              void* d_out, int out_size, const int* in_dtypes, int out_dtype,
                              void* const* d_out_all, void* const* d_ws_all, size_t ws_bytes) {
  if (nranks != NR || (unsigned)rank >= (unsigned)NR) return;
  if (ws_bytes < WS_PKT_OFF + PKT_REGION) return;
  static bool init[NR] = {};
  if (!init[rank]) {
    HIP_CHECK(hipFuncSetAttribute((const void*)fused_kernel, hipFuncAttributeMaxDynamicSharedMemorySize, SMEM));
    init[rank] = true;
  }
  Args a;
  a.x = (const char*)d_in[0]; a.w = (const char*)d_in[1]; a.sx = (const float*)d_in[2]; a.sw = (const float*)d_in[3];
  for (int p = 0; p < NR; ++p) { a.out[p] = (float*)d_out_all[p]; a.ws[p] = (u32*)d_ws_all[p]; }
  a.rank = rank;
  fused_kernel<<<NWG, NTHR, SMEM, stream>>>(a);
}
```
